```python
import numpy as np

kernel_name = "gpu_ag_gemm_m4096_k4096_n32768_bf16_gelu"


ABSMAX_RTOL = 0.008

COMPUTE_DTYPE = "bfloat16"
EPILOGUE = "gelu"
IN_QUANT = None
OUT_QUANT = None
IN_DTYPES = {"x": "bfloat16", "w_mat": "bfloat16"}
WORLD_SIZE = 8

def _to_bf16(x):
    u = np.ascontiguousarray(x, dtype=np.float32).view(np.uint32)
    return ((u + 0x7FFF + ((u >> 16) & 1)) & 0xFFFF0000).view(np.float32)

def setup_inputs():

    x = _to_bf16(np.random.randn(4096, 4096).astype(np.float32))
    w_mat = _to_bf16((np.random.randn(4096, 32768) / np.sqrt(4096)).astype(np.float32))
    return {"x": x, "w_mat": w_mat}

def _epilogue(y):
    c = 0.7978845608028654
    return (0.5 * y * (1.0 + np.tanh(c * (y + 0.044715 * y ** 3)))).astype(np.float32)

def reference(x, w_mat):
    y = (x @ w_mat).astype(np.float32)
    return _epilogue(y)
```

```cpp
#include <hip/hip_runtime.h>
#include <rccl/rccl.h>
#include <cstdio>

#define HIP_CHECK(expr) do { hipError_t _e = (expr); if (_e != hipSuccess) { \
    fprintf(stderr, "HIP error %s at %s:%d\n", hipGetErrorString(_e), __FILE__, __LINE__); \
    return; } } while (0)

#define LAS __attribute__((address_space(3)))
typedef unsigned short u16;
typedef short bf16x8 __attribute__((ext_vector_type(8)));
typedef short s16x4 __attribute__((ext_vector_type(4)));
typedef float f32x4 __attribute__((ext_vector_type(4)));
typedef float f32x2 __attribute__((ext_vector_type(2)));
typedef __bf16 bf16x2 __attribute__((ext_vector_type(2)));
typedef unsigned u32x4 __attribute__((ext_vector_type(4)));
typedef unsigned u32x2 __attribute__((ext_vector_type(2)));

constexpr int NR = 8, M = 4096, N = 4096, K = 4096;
constexpr int BM = 256, BK = 64, HALF = 128, HTB = HALF * BK * 2  , LDS_BYTES = 8 * HTB;
constexpr int LOCALX = 1;
constexpr size_t XS_BYTES = (size_t)(M / NR) * K * 2;
constexpr size_t HB_OFF = 2 * XS_BYTES;
constexpr size_t HB_BYTES = (size_t)16 * 4 * HTB;
constexpr size_t FLAG_OFF = HB_OFF + 2 * HB_BYTES;
constexpr size_t HFLAG_OFF = FLAG_OFF + 16 * 512;
constexpr size_t CNT_OFF = HFLAG_OFF + 16 * 8 * 128;
constexpr size_t TMO_OFF = CNT_OFF + 256 * 128;
constexpr size_t TICK_OFF = TMO_OFF + 128;
constexpr size_t WS_NEED = TICK_OFF + 8 * 128;
constexpr unsigned long long DEADLINE_TICKS = 100000000ull;

struct Params {
    const u16* x; const u16* w; u16* out;
    char* ws[NR];
    unsigned srcmap;
    int rank;
};

__device__ __forceinline__ int lds_byte(int r, int c) { const int st = (r >> 4) * 2 + (c >> 5), rr = r & 15, cc = c & 31, ob = rr * 64 + cc * 2; return st * 1024 + (ob ^ (((ob >> 9) & 1) << 5)); }
__device__ __forceinline__ unsigned gelu2(float y0, float y1) {
    const float k0 = -2.0f * 0.7978845608028654f * 1.4426950408889634f, k1 = k0 * 0.044715f;
    const f32x2 y = {y0, y1};
    const f32x2 pl = __builtin_elementwise_fma(y * y, (f32x2){k1, k1}, (f32x2){k0, k0});
    const f32x2 z = y * pl;
    const f32x2 d = (f32x2){__builtin_amdgcn_exp2f(z[0]), __builtin_amdgcn_exp2f(z[1])} + 1.0f;
    const f32x2 o = y * (f32x2){__builtin_amdgcn_rcpf(d[0]), __builtin_amdgcn_rcpf(d[1])};
    return __builtin_bit_cast(unsigned, __builtin_convertvector(o, bf16x2));
}
#define WAIT_V(n) asm volatile("s_waitcnt vmcnt(" #n ")" ::: "memory")
#define WAIT_L(n) asm volatile("s_waitcnt lgkmcnt(" #n ")" ::: "memory")
#define LGKM0 __builtin_amdgcn_s_waitcnt(0xC07F)
#define BAR __builtin_amdgcn_s_barrier()
#define SCHED __builtin_amdgcn_sched_barrier(0)
#define RFL(x) ((unsigned)__builtin_amdgcn_readfirstlane((int)(x)))

#define SA(b, h) (((b) * 2 + (h)) * HTB)
#define SB(b, h) ((4 + (b) * 2 + (h)) * HTB)

__global__ __launch_bounds__(512, 2) void ag_gemm(const Params p) {
    extern __shared__ __attribute__((aligned(16))) unsigned char shm_[];
    LAS unsigned char* lds = (LAS unsigned char*)shm_;
    const int tid = threadIdx.x, wid = __builtin_amdgcn_readfirstlane(tid >> 6), lane = tid & 63, wr = wid >> 2, wc = wid & 3, fr = lane & 15, fq = lane >> 4;
    constexpr int nt = K / BK;
    const int g = blockIdx.x;
    const int rank = p.rank;
    char* const myws = p.ws[LOCALX];
    unsigned xcc; asm volatile("s_getreg_b32 %0, hwreg(HW_REG_XCC_ID)" : "=s"(xcc)); xcc &= 7u;

    const int pt = g >> 2, pr = (g >> 1) & 1, ph = g & 1;
    const bool hpush = g < 64 && (g & 7) != LOCALX;
    const int hu = (g >> 4) & 3, hr = (g >> 3) & 1;
    char* const hdst = p.ws[g & 7];
    const int sw = (lane * 16) ^ ((((lane * 16) >> 9) & 1) << 5), srr = sw >> 6, scc = (sw & 63) >> 1;
    u32x4 xd0, xd1, hd0, hd1; unsigned cnt, tick = 0;
    if (wid == 0) {
        unsigned long long ex_; unsigned one = 1, toff = xcc * 128; const char* tptr = myws + TICK_OFF;
        asm volatile("s_mov_b64 %1, exec\n\ts_mov_b64 exec, 1\n\tglobal_atomic_add %0, %2, %3, %4 sc0\n\ts_mov_b64 exec, %1"
                     : "=&v"(tick), "=&s"(ex_) : "v"(toff), "v"(one), "s"(tptr) : "memory");
    }
    {
        const unsigned hoff = (unsigned)(((hr * 256 + (hu & 1) * 128 + wid * 16 + srr) * K + (hu >> 1) * 64 + scc) * 2);
        const unsigned soff = (unsigned)(((pr * 256 + ph * 128 + wid * 16 + srr) * K + pt * 64 + scc) * 2);
        const char* cptr = myws + CNT_OFF + (size_t)g * 128;
        unsigned zero = 0;
        asm volatile("global_load_dword %4, %8, %9 sc0 sc1\n\t"
                     "global_load_dwordx4 %2, %5, %7\n\tglobal_load_dwordx4 %3, %5, %7 offset:64\n\t"
                     "global_load_dwordx4 %0, %6, %7\n\tglobal_load_dwordx4 %1, %6, %7 offset:64"
                     : "=&v"(xd0), "=&v"(xd1), "=&v"(hd0), "=&v"(hd1), "=&v"(cnt) : "v"(hoff), "v"(soff), "s"(p.x), "v"(zero), "s"(cptr) : "memory");
    }
    unsigned long long fa;
    {
        int lo = 0, hi = 0;
#pragma unroll
        for (int q = 0; q < 7; ++q) {
            const unsigned long long a = (unsigned long long)p.ws[q < LOCALX ? q : q + 1] + FLAG_OFF + (size_t)(rank * 2 + pr) * 512 + (2 * pt + ph) * 4;
            asm volatile("v_writelane_b32 %0, %2, %4\n\tv_writelane_b32 %1, %3, %4" : "+v"(lo), "+v"(hi) : "s"((unsigned)a), "s"((unsigned)(a >> 32)), "i"(q));
        }
        fa = ((unsigned long long)(unsigned)hi << 32) | (unsigned)lo;
    }
    asm volatile("s_waitcnt vmcnt(2)" : "+v"(hd0), "+v"(hd1), "+v"(cnt), "+v"(fa), "+v"(tick) :: "memory");
    const unsigned seq = RFL(cnt) + 1u;
    const unsigned par = seq & 1u;
    const unsigned hdoff = (unsigned)(HB_OFF + par * HB_BYTES + (size_t)((rank * 2 + hr) * 4 + hu) * HTB + wid * 2048 + lane * 16);
    const unsigned hfoff = (unsigned)(HFLAG_OFF + (size_t)((rank * 2 + hr) * 8 + wid) * 128 + hu * 4);
#define HEAD_STORES() asm volatile("global_store_dwordx4 %0, %1, %2 sc0 sc1\n\tglobal_store_dwordx4 %0, %3, %2 offset:1024 sc0 sc1\n\ts_nop 1" \
                     :: "v"(hdoff), "v"(hd0), "s"(hdst), "v"(hd1) : "memory")
#define HEAD_FLAG() do { unsigned long long ex_; \
        asm volatile("s_mov_b64 %0, exec\n\ts_mov_b64 exec, 1\n\tglobal_store_dword %1, %2, %3 sc0 sc1\n\ts_mov_b64 exec, %0\n\ts_nop 1" \
                     : "=&s"(ex_) : "v"(hfoff), "v"(seq), "s"(hdst) : "memory"); } while (0)
    if (hpush && hu < 2) HEAD_STORES();
    asm volatile("s_waitcnt vmcnt(2)" : "+v"(xd0), "+v"(xd1) :: "memory");
    if (!(hpush && hu < 2)) asm volatile("s_waitcnt vmcnt(0)" : "+v"(xd0), "+v"(xd1) :: "memory");
    {
        const unsigned doff = (unsigned)(par * XS_BYTES + (size_t)((pr * 64 + pt) * 2 + ph) * HTB + wid * 2048 + lane * 16);
        asm volatile("global_store_dwordx4 %0, %1, %2 sc0 sc1\n\tglobal_store_dwordx4 %0, %3, %2 offset:1024 sc0 sc1\n\ts_nop 1"
                     :: "v"(doff), "v"(xd0), "s"(myws), "v"(xd1) : "memory");
    }
    if (hpush && hu >= 2) HEAD_STORES();
    if (wid == 0) {
        unsigned grp = xcc, slot = RFL(tick) - 32u * (seq - 1u), tries = 0;
        while (slot >= 32u) {
            if (++tries > 4096u) { __hip_atomic_store((unsigned*)(myws + TMO_OFF), seq, __ATOMIC_RELAXED, __HIP_MEMORY_SCOPE_AGENT); slot = 0; break; }
            unsigned* tp = (unsigned*)(myws + TICK_OFF);
            unsigned nv = 0;
            if (lane == 0) { __hip_atomic_fetch_sub(tp + grp * 32, 1u, __ATOMIC_RELAXED, __HIP_MEMORY_SCOPE_AGENT);
                             nv = __hip_atomic_fetch_add(tp + ((grp + 1) & 7) * 32, 1u, __ATOMIC_RELAXED, __HIP_MEMORY_SCOPE_AGENT); }
            grp = (grp + 1) & 7; slot = RFL(nv) - 32u * (seq - 1u);
        }
        *(LAS unsigned*)(lds + SA(1, 1)) = grp * 32 + slot;
        WAIT_L(0);
    }
    BAR;
    const unsigned tile = RFL(*(volatile LAS unsigned*)(lds + SA(1, 1)));
    const int grp = tile >> 5, src = (p.srcmap >> (grp * 4)) & 7, tr = tile & 1, pm = src * 2 + tr, pn = (tile >> 1) & 15;
    const bool local = grp == LOCALX;
    char* const srcws = p.ws[grp];
    unsigned voffA0, voffA1, voffB[2];
    if (local) { voffA0 = (unsigned)((wid * 16 + srr) * K + scc) * 2u; voffA1 = voffA0 + 64u; }
    else { voffA0 = (unsigned)(wid * 2048 + lane * 16); voffA1 = voffA0 + 1024u; }
#pragma unroll
    for (int i = 0; i < 2; ++i) {
        const int k = i * 32 + wid * 4 + (lane >> 4), hk = (k & 3) | (((k >> 3) & 1) << 2), ch = (lane & 15) ^ (hk << 1);
        voffB[i] = (unsigned)(k * N + ch * 8) * 2u;
    }
    const size_t kstepA = local ? (size_t)(BK * 2) : (size_t)(2 * HTB);
    const size_t hstepA = local ? (size_t)HALF * K * 2 : (size_t)HTB;
    const size_t kstepB = (size_t)BK * N * 2, hstepB = HALF * 2;
    const int aoff = lds_byte(wr * 64 + fr, fq * 8);
    int boff[2];
    { const int gq = lane >> 4, q = (lane >> 2) & 3, pp = lane & 3, h = q | ((gq & 1) << 2);
#pragma unroll
      for (int n = 0; n < 2; ++n) boff[n] = (int)(unsigned)(size_t)lds + 4 * HTB + (8 * gq + q) * 256 + (((wc * 2 + n) ^ h) * 32) + pp * 8; }
#define STA(bufoff, gbase) do { \
        __builtin_amdgcn_global_load_lds((const unsigned*)((const char*)(gbase) + voffA0), (LAS unsigned*)(lds + (bufoff) + wid * 2048), 16, 0, 0); \
        __builtin_amdgcn_global_load_lds((const unsigned*)((const char*)(gbase) + voffA1), (LAS unsigned*)(lds + (bufoff) + wid * 2048 + 1024), 16, 0, 0); } while (0)
#define STB(bufoff, gbase) do { _Pragma("unroll") for (int _i = 0; _i < 2; ++_i) \
        __builtin_amdgcn_global_load_lds((const unsigned*)((const char*)(gbase) + voffB[_i]), (LAS unsigned*)(lds + (bufoff) + wid * 1024 + _i * 8192), 16, 0, 0); } while (0)
#define LDA(dst, b, h) do { _Pragma("unroll") for (int m = 0; m < 4; ++m) _Pragma("unroll") for (int k = 0; k < 2; ++k) dst[m][k] = *(const LAS bf16x8*)(lds + SA(b, h) + aoff + m * 2048 + k * 1024); } while (0)
#define TRRD(dst, addr, off) asm volatile("ds_read_b64_tr_b16 %0, %1 offset:%2" : "=v"(dst) : "v"(addr), "i"(off) : "memory")
#define LDB(dst, b, h) do { _Pragma("unroll") for (int n = 0; n < 2; ++n) { \
        TRRD(dst##l[n][0], boff[n], SB(b, h) - SB(0, 0)); TRRD(dst##h_[n][0], boff[n], SB(b, h) - SB(0, 0) + 1024); \
        TRRD(dst##l[n][1], boff[n], SB(b, h) - SB(0, 0) + 8192); TRRD(dst##h_[n][1], boff[n], SB(b, h) - SB(0, 0) + 8192 + 1024); } } while (0)
#define PINB(dst) asm volatile("" : "+v"(dst##l[0][0]), "+v"(dst##l[0][1]), "+v"(dst##l[1][0]), "+v"(dst##l[1][1]), "+v"(dst##h_[0][0]), "+v"(dst##h_[0][1]), "+v"(dst##h_[1][0]), "+v"(dst##h_[1][1]))
#define BFRAG(Bt, n, k) __builtin_shufflevector(Bt##l[n][k], Bt##h_[n][k], 0, 1, 2, 3, 4, 5, 6, 7)
#define MF(ai, bj, m, n, k, Bt) do { acc[ai][bj][m][n] = __builtin_amdgcn_mfma_f32_16x16x32_bf16(BFRAG(Bt, n, k), At[m][k], acc[ai][bj][m][n], 0, 0, 0); SCHED; } while (0)
#define MFJ(ai, m, j, k) do { if ((j) < 2) MF(ai, 0, m, (j) & 1, k, B0); else MF(ai, 1, m, (j) & 1, k, B1); } while (0)
#define MMA2(ai) do { __builtin_amdgcn_s_setprio(1); \
        _Pragma("unroll") for (int m = 0; m < 4; ++m) _Pragma("unroll") for (int jj = 0; jj < 4; ++jj) _Pragma("unroll") for (int k = 0; k < 2; ++k) \
            MFJ(ai, m, ((m & 1) ? 3 - jj : jj), ((jj & 1) ? 1 - k : k)); \
        __builtin_amdgcn_s_setprio(0); } while (0)

    const char* cB = (const char*)p.w + (size_t)pn * 256 * 2;
    const char* cA = local ? (const char*)p.x + (size_t)tr * 256 * K * 2 : (const char*)srcws + par * XS_BYTES + (size_t)tr * 256 * K * 2;
    const char* hA = local ? cA : (const char*)myws + HB_OFF + par * HB_BYTES + (size_t)pm * 4 * HTB;
    unsigned* const tmo = (unsigned*)(myws + TMO_OFF);
    STB(SB(0, 0), cB); STB(SB(0, 1), cB + hstepB);
    if (hpush && hu < 2) { WAIT_V(6); HEAD_FLAG(); }
    if (local) { STA(SA(0, 0), hA); STA(SA(0, 1), hA + hstepA); }
    STB(SB(1, 0), cB + kstepB); STB(SB(1, 1), cB + hstepB + kstepB);
    if (local) WAIT_V(12); else WAIT_V(8);
    if (hpush && hu >= 2) HEAD_FLAG();
    BAR;
    if (wid == 0) {
        unsigned long long ex_;
        asm volatile("s_mov_b64 %0, exec\n\ts_mov_b64 exec, 0x7f\n\tglobal_store_dword %1, %2, off sc0 sc1\n\ts_mov_b64 exec, %0\n\ts_nop 1"
                     : "=&s"(ex_) : "v"(fa), "v"(seq) : "memory");
    }
    f32x4 acc[2][2][4][2];
#pragma unroll
    for (int a = 0; a < 2; ++a)
#pragma unroll
        for (int b = 0; b < 2; ++b)
#pragma unroll
            for (int m = 0; m < 4; ++m)
#pragma unroll
                for (int n = 0; n < 2; ++n) { acc[a][b][m][n] = (f32x4){0.f, 0.f, 0.f, 0.f}; asm volatile("" : "+v"(acc[a][b][m][n])); }
    unsigned need = local ? 0u : seq;
    unsigned f0 = 0, f1 = 0, f2 = 0, f3 = 0;
    const char* hfbase = myws + HFLAG_OFF + (size_t)(pm * 8 + wid) * 128;
    const char* fbase = myws + FLAG_OFF + (size_t)pm * 512;
#define GIVEUP(ts_) ((++np_ & 31u) == 0u && (RFL(__hip_atomic_load(tmo, __ATOMIC_RELAXED, __HIP_MEMORY_SCOPE_AGENT)) == seq || __builtin_amdgcn_s_memrealtime() - (ts_) > DEADLINE_TICKS))
#define HWAIT(W) do { if (W < need) { \
        const unsigned long long ts_ = __builtin_amdgcn_s_memrealtime(); unsigned np_ = 0; \
        for (;;) { \
            u32x4 tf_; unsigned z_ = 0; \
            asm volatile("global_load_dwordx4 %0, %1, %2 sc0 sc1\n\ts_waitcnt vmcnt(0)" : "=&v"(tf_) : "v"(z_), "s"(hfbase) : "memory"); \
            f0 = RFL(tf_[0]); f1 = RFL(tf_[1]); f2 = RFL(tf_[2]); f3 = RFL(tf_[3]); \
            if (W >= need) break; \
            if (GIVEUP(ts_)) { __hip_atomic_store(tmo, seq, __ATOMIC_RELAXED, __HIP_MEMORY_SCOPE_AGENT); need = 0; WAIT_V(0); break; } \
            __builtin_amdgcn_s_sleep(1); \
        } LGKM0; } } while (0)
    if (!local) {
        HWAIT(f0); STA(SA(0, 0), hA);
        HWAIT(f1); STA(SA(0, 1), hA + hstepA);
        WAIT_V(2);
    } else WAIT_V(6);
    bf16x8 At[4][2]; s16x4 B0l[2][2], B0h_[2][2], B1l[2][2], B1h_[2][2];
    LGKM0; BAR;
    if (wr == 1) BAR;
    bool bad;
    {
        u32x2 tf;
        HWAIT(f2);
        LDB(B0, 0, 0); LDB(B1, 0, 1); SCHED; LDA(At, 0, 0); STA(SA(1, 0), hA + kstepA);
        { unsigned go = lane * 8; asm volatile("global_load_dwordx2 %0, %1, %2 sc0 sc1" : "=v"(tf) : "v"(go), "s"(fbase) : "memory"); }
        WAIT_V(3); WAIT_L(0); PINB(B0); PINB(B1); BAR; MMA2(0); BAR; SCHED;
        HWAIT(f3);
        LDA(At, 0, 1); STA(SA(1, 1), hA + kstepA + hstepA); STB(SB(0, 0), cB + 2 * kstepB); STB(SB(0, 1), cB + 2 * kstepB + hstepB);
        asm volatile("s_waitcnt vmcnt(6)" : "+v"(tf) :: "memory");
        if (need && !__all(tf[0] >= need && tf[1] >= need)) {
            const unsigned long long ts_ = __builtin_amdgcn_s_memrealtime(); unsigned np_ = 0;
            for (;;) {
                unsigned go = lane * 8;
                asm volatile("global_load_dwordx2 %0, %1, %2 sc0 sc1\n\ts_waitcnt vmcnt(0)" : "=&v"(tf) : "v"(go), "s"(fbase) : "memory");
                if (__all(tf[0] >= need && tf[1] >= need)) break;
                if (GIVEUP(ts_)) { __hip_atomic_store(tmo, seq, __ATOMIC_RELAXED, __HIP_MEMORY_SCOPE_AGENT); WAIT_V(0); break; }
                __builtin_amdgcn_s_sleep(1);
            }
            LGKM0;
        }
        STA(SA(0, 0), cA + 2 * kstepA);
        WAIT_L(0); BAR; MMA2(1); BAR; SCHED;
        unsigned tmov;
        { unsigned zero = 0; asm volatile("global_load_dword %0, %1, %2 sc1" : "=v"(tmov) : "v"(zero), "s"(tmo) : "memory"); }
        LDB(B0, 1, 0); LDB(B1, 1, 1); SCHED; LDA(At, 1, 0); STA(SA(0, 1), cA + 2 * kstepA + hstepA);
        WAIT_V(9); WAIT_L(0); PINB(B0); PINB(B1); BAR; MMA2(0); BAR; SCHED;
        LDA(At, 1, 1); STB(SB(1, 0), cB + 3 * kstepB); STB(SB(1, 1), cB + 3 * kstepB + hstepB); STA(SA(1, 0), cA + 3 * kstepA);
        asm volatile("s_waitcnt vmcnt(8)" : "+v"(tmov) :: "memory");
        bad = RFL(tmov) == seq;
        WAIT_L(0); BAR; MMA2(1); BAR; SCHED;
    }
    for (int t = 2; t < nt - 2; t += 2) {
        const char* a1 = cA + (size_t)(t + 1) * kstepA;
        const char* a2 = cA + (size_t)(t + 2) * kstepA; const char* b2 = cB + (size_t)(t + 2) * kstepB;
        const char* a3 = a2 + kstepA; const char* b3 = b2 + kstepB;
        LDB(B0, 0, 0); LDB(B1, 0, 1); SCHED; LDA(At, 0, 0); STA(SA(1, 1), a1 + hstepA);
        WAIT_V(8); WAIT_L(0); PINB(B0); PINB(B1); BAR; MMA2(0); BAR; SCHED;
        LDA(At, 0, 1); STB(SB(0, 0), b2); STB(SB(0, 1), b2 + hstepB); STA(SA(0, 0), a2);
        WAIT_V(8); WAIT_L(0); BAR; MMA2(1); BAR; SCHED;
        LDB(B0, 1, 0); LDB(B1, 1, 1); SCHED; LDA(At, 1, 0); STA(SA(0, 1), a2 + hstepA);
        WAIT_V(8); WAIT_L(0); PINB(B0); PINB(B1); BAR; MMA2(0); BAR; SCHED;
        LDA(At, 1, 1); STB(SB(1, 0), b3); STB(SB(1, 1), b3 + hstepB); STA(SA(1, 0), a3);
        WAIT_V(8); WAIT_L(0); BAR; MMA2(1); BAR; SCHED;
    }
    int tz = threadIdx.x; asm volatile("" : "+v"(tz));
    u16* const obase = p.out + (size_t)(pm * BM + wr * 64 + (tz & 15)) * N + (pn * BM + wc * 32 + (tz & 16) + ((tz & 32) >> 2));
#define EPI(ai, m) do { _Pragma("unroll") for (int bj = 0; bj < 2; ++bj) { \
        const f32x4 v0 = acc[ai][bj][m][0], v1 = acc[ai][bj][m][1]; \
        const unsigned a0 = gelu2(v0[0], v0[1]), a1 = gelu2(v0[2], v0[3]), c0 = gelu2(v1[0], v1[1]), c1 = gelu2(v1[2], v1[3]); \
        const auto r0 = __builtin_amdgcn_permlane16_swap(a0, c0, false, false); \
        const auto r1 = __builtin_amdgcn_permlane16_swap(a1, c1, false, false); \
        const u32x4 w = {r0[0], r1[0], r0[1], r1[1]}; \
        asm volatile("global_store_dwordx4 %0, %1, off offset:%2 sc1\n\ts_nop 1" :: "v"(obase + (size_t)((ai) * HALF + (m) * 16) * N), "v"(w), "i"(bj * HALF * 2) : "memory"); } } while (0)
    LDB(B0, 0, 0); LDB(B1, 0, 1); SCHED; LDA(At, 0, 0); STA(SA(1, 1), cA + (size_t)(nt - 1) * kstepA + hstepA);
    WAIT_V(2); WAIT_L(0); PINB(B0); PINB(B1); BAR; MMA2(0); BAR; SCHED;
    LDB(B0, 1, 0); LDB(B1, 1, 1); SCHED; LDA(At, 1, 0);
    WAIT_V(0); WAIT_L(0); PINB(B0); PINB(B1); BAR; MMA2(0); if (wr == 0) BAR; SCHED;
    if (wid == 0 && lane == 0) __hip_atomic_store((unsigned*)(myws + CNT_OFF + (size_t)g * 128), seq, __ATOMIC_RELAXED, __HIP_MEMORY_SCOPE_AGENT);
    LDA(At, 1, 1); SCHED;
    EPI(0, 0); EPI(0, 1); SCHED;
    MMA2(1); SCHED;
    LDB(B0, 0, 0); LDB(B1, 0, 1); LDA(At, 0, 1); SCHED;
    EPI(0, 2); EPI(0, 3); SCHED;
    WAIT_L(0); PINB(B0); PINB(B1);
    MMA2(1); SCHED;
    EPI(1, 0); EPI(1, 1); EPI(1, 2); EPI(1, 3);
    if (__builtin_expect(bad, 0)) {
        WAIT_V(0);
        const u32x4 w = {0x7fc07fc0u, 0x7fc07fc0u, 0x7fc07fc0u, 0x7fc07fc0u};
        for (int i = 0; i < 16; ++i) *(u32x4*)(obase + (size_t)((i >> 3) * HALF + ((i >> 1) & 3) * 16) * N + (i & 1) * HALF) = w;
    }
    WAIT_V(0);
}

extern "C" void kernel_launch(int rank, int nranks, ncclComm_t comm,
                              hipStream_t stream,
                              void* const* d_in, const int* in_sizes, int n_in,
                              void* d_out, int out_size,
                              const int* in_dtypes, int out_dtype,
                              void* const* d_out_all,
                              void* const* d_ws_all, size_t ws_bytes) {
    static bool init = false;
    static unsigned quad = 0xB2;
    if (!init) {
        HIP_CHECK(hipFuncSetAttribute((const void*)ag_gemm, hipFuncAttributeMaxDynamicSharedMemorySize, LDS_BYTES));
        int dev = -1, nd = 0, bus[NR]; bool ok = hipGetDevice(&dev) == hipSuccess && hipGetDeviceCount(&nd) == hipSuccess && dev == rank && nd == NR;
        for (int i = 0; ok && i < NR; ++i) ok = hipDeviceGetAttribute(&bus[i], hipDeviceAttributePciBusId, i) == hipSuccess;
        if (ok) {
            unsigned q = 0, seen = 0;
            for (int i = 0; i < NR; ++i) { int pos = 0; for (int j = 0; j < NR; ++j) pos += bus[j] < bus[i]; seen |= 1u << pos; q |= ((0xB1u >> pos) & 1u) << i; }
            if (seen == 0xFFu) quad = q;
        }
        (void)hipGetLastError();
        init = true;
    }
    if (nranks != NR || (unsigned)rank >= (unsigned)NR || ws_bytes < WS_NEED || n_in < 2 ||
        in_sizes[0] != (M / NR) * K || in_sizes[1] != K * N || out_size != M * N) { fprintf(stderr, "unexpected configuration\n"); return; }
    Params p;
    p.x = (const u16*)d_in[0]; p.w = (const u16*)d_in[1]; p.out = (u16*)d_out; p.rank = rank;
    int order[NR], no = 0;
    order[no++] = rank;
    for (int i = 1; i < NR; ++i) { const int r = (rank + i) % NR; if (((quad >> r) & 1) == ((quad >> rank) & 1)) order[no++] = r; }
    for (int i = 1; i < NR; ++i) { const int r = (rank + i) % NR; if (((quad >> r) & 1) != ((quad >> rank) & 1)) order[no++] = r; }
    { const int tmp = order[0]; order[0] = order[LOCALX]; order[LOCALX] = tmp; }
    p.srcmap = 0;
    for (int i = 0; i < NR; ++i) { p.ws[i] = (char*)d_ws_all[order[i]]; p.srcmap |= (unsigned)order[i] << (4 * i); }
    hipLaunchKernelGGL(ag_gemm, dim3(256), dim3(512), LDS_BYTES, stream, p);
    HIP_CHECK(hipGetLastError());
}
```
